```python
import numpy as np

kernel_name = "gpu_ag_gemm_m4096_k4096_n4096_bf16_q8"


ABSMAX_RTOL = 0.02

COMPUTE_DTYPE = "bfloat16"
EPILOGUE = "quant_int8"
IN_QUANT = None
OUT_QUANT = "int8"
IN_DTYPES = {"x": "bfloat16", "w_mat": "bfloat16"}
WORLD_SIZE = 8

def _to_bf16(x):
    u = np.ascontiguousarray(x, dtype=np.float32).view(np.uint32)
    return ((u + 0x7FFF + ((u >> 16) & 1)) & 0xFFFF0000).view(np.float32)

def setup_inputs():

    x = _to_bf16(np.random.randn(4096, 4096).astype(np.float32))
    w_mat = _to_bf16((np.random.randn(4096, 4096) / np.sqrt(4096)).astype(np.float32))
    return {"x": x, "w_mat": w_mat}

def _quant_dequant(y, amax):
    scale = amax / 127.0
    q = np.clip(np.round(y / scale), -127.0, 127.0)
    return (q * scale).astype(np.float32)
def _epilogue(y):
    return _quant_dequant(y, np.abs(y).max())

def reference(x, w_mat):
    y = (x @ w_mat).astype(np.float32)
    return _epilogue(y)
```

```cpp
#include <hip/hip_runtime.h>
#include <rccl/rccl.h>
#include <cstdio>
#include <cstdint>

typedef unsigned int u32;
typedef unsigned long long u64;
typedef __attribute__((ext_vector_type(4))) unsigned int u32x4;
typedef __attribute__((ext_vector_type(2))) unsigned int u32x2;
typedef __attribute__((ext_vector_type(4))) float f32x4;
typedef __attribute__((ext_vector_type(8))) short s16x8;
typedef __attribute__((ext_vector_type(4))) short s16x4;
#define LAS __attribute__((address_space(3)))
#define GAS __attribute__((address_space(1)))

constexpr int NR = 8;
constexpr int KDIM = 4096;
constexpr int NSTEP = 64;
constexpr int RING_B = 6 * 24576;
constexpr int NWG = 256;
#define PWIN 2
constexpr int PW = PWIN;
constexpr int NPIECE = 80;
constexpr int NREP = 32;
constexpr u64 DEADLINE_TICKS = 60000000ull;

constexpr u32 WS_EPOCH = 0;
#define FSTRIDE 128
constexpr u32 WS_DFLAG = 65536;
constexpr u32 WS_AMAX1 = 8192;
constexpr u32 WS_AMAX2 = 16384;
constexpr u32 TMO_OFF = 64;
constexpr u32 WS_OFLAG = 32768;
constexpr u32 WS_DBG = 262144;
constexpr u32 WS_X = 1u << 20;
constexpr u32 XSLOT_B = 512u << 10;
constexpr u32 WS_W = WS_X + 32 * XSLOT_B;
constexpr u32 WSLOT_B = 1u << 20;
constexpr u32 WS_END = WS_W + 4 * WSLOT_B;
constexpr u32 IN_B = 4u << 20;
constexpr u32 OUT_B = 2u << 20;

constexpr int C_DONE = 0, C_READY = 8, C_AVX = 12, C_AVW = 13, C_WMAX = 14, C_WCNT = 15, C_GMAX = 16, C_GRDY = 17, C_CALL = 18;

struct Params {
    const char* x;
    const char* w;
    char* ws[NR];
    char* out[NR];
    int rank;
    u32 tile[NWG];
};
constexpr u32 T_SRANK = 0, T_PANEL = 3, T_SLOT = 6, T_WLOCAL = 11, T_CI = 12, T_DBL = 14, T_PANELB = 15, T_PUSH = 18;

__device__ __forceinline__ u32x4 make_srd(const void* base, u32 bytes) {
    u64 a = (u64)base;
    u32x4 r;
    r.x = __builtin_amdgcn_readfirstlane((u32)a);
    r.y = __builtin_amdgcn_readfirstlane((u32)(a >> 32) & 0xffffu);
    r.z = bytes;
    r.w = 0x00020000u;
    return r;
}
__device__ __forceinline__ u32 ld_sys(const void* p) { return __hip_atomic_load((GAS const u32*)p, __ATOMIC_RELAXED, __HIP_MEMORY_SCOPE_SYSTEM); }
__device__ __forceinline__ u64 ld_sys64(const void* p) { return __hip_atomic_load((GAS const u64*)p, __ATOMIC_RELAXED, __HIP_MEMORY_SCOPE_SYSTEM); }
__device__ __forceinline__ void st_sys(void* p, u32 v) { __hip_atomic_store((GAS u32*)p, v, __ATOMIC_RELAXED, __HIP_MEMORY_SCOPE_SYSTEM); }
__device__ __forceinline__ void st_agent(void* p, u32 v) { __hip_atomic_store((GAS u32*)p, v, __ATOMIC_RELAXED, __HIP_MEMORY_SCOPE_AGENT); }
__device__ __forceinline__ u64 now() { return __builtin_amdgcn_s_memrealtime(); }
__device__ __forceinline__ u32 rfl(u32 v) { return __builtin_amdgcn_readfirstlane(v); }
#define POLL_SLEEP 4
#define OUTMODE 0
#define POLL_STAG 6
#define NAGG 1
#define POLLBITS "sc0 sc1"
#define FDENSE 11
#define DMABITS "sc0 sc1"
#define CFENCE() asm volatile("" ::: "memory")
#define VMWAIT(n) asm volatile("s_waitcnt vmcnt(" #n ")" ::: "memory")

#define STAMP(i) do {} while (0)
#define STAMPV(i, v) do {} while (0)

template <int CTRL> __device__ __forceinline__ u32 dpp(u32 v) { return __builtin_amdgcn_update_dpp(v, v, CTRL, 0xf, 0xf, false); }
__device__ __forceinline__ u32 row16_max(u32 v) {
    v = max(v, dpp<0xB1>(v)); v = max(v, dpp<0x4E>(v)); v = max(v, dpp<0x141>(v)); v = max(v, dpp<0x140>(v)); return v;
}
__device__ __forceinline__ u32 wave_max(u32 v) {
    v = row16_max(v);
    return max(max((u32)__builtin_amdgcn_readlane(v, 0), (u32)__builtin_amdgcn_readlane(v, 16)),
               max((u32)__builtin_amdgcn_readlane(v, 32), (u32)__builtin_amdgcn_readlane(v, 48)));
}

__device__ __forceinline__ u32 xsrc_off(int panel, int piece, int lane) {
    int r = piece * 8 + (lane >> 3);
    return (u32)(panel * 64 + r) * (KDIM * 2) + (u32)(((lane & 7) ^ ((r >> 1) & 7)) << 4);
}
__device__ __forceinline__ u32 wsrc_off(int blk, int piece, int lane) {
    int k = piece * 4 + (lane >> 4);
    return (u32)k * 1024u + (u32)blk * 256u + (u32)(((lane & 15) ^ ((k & 3) | (((k >> 3) & 1) << 3))) << 4);
}

__device__ __forceinline__ void dma4(u32 lds, u32 v0, u32 v1, u32 v2, u32 v3, u32x4 srd, u32 soff) {
    u32 keep;
    asm volatile(
        "s_mov_b32 %0, m0\n\t"
        "s_mov_b32 m0, %1\n\t"
        "s_nop 0\n\t"
        "buffer_load_dwordx4 %2, %6, %7 offen " DMABITS " lds\n\t"
        "buffer_load_dwordx4 %3, %6, %7 offen offset:1024 " DMABITS " lds\n\t"
        "buffer_load_dwordx4 %4, %6, %7 offen offset:2048 " DMABITS " lds\n\t"
        "buffer_load_dwordx4 %5, %6, %7 offen offset:3072 " DMABITS " lds\n\t"
        "s_mov_b32 m0, %0"
        : "=&s"(keep)
        : "s"(lds), "v"(v0), "v"(v1), "v"(v2), "v"(v3), "s"(srd), "s"(soff)
        : "memory");
}

__device__ __forceinline__ void raise_tmo(char* ws, u32 call, int lane, u32 site = 0) {
    if (lane < NREP) st_agent(ws + WS_AMAX2 + lane * 128 + TMO_OFF, call);
}
#define POLL_ISSUE32(v, addr) asm volatile("global_load_dword %0, %1, off " POLLBITS : "+v"(v) : "v"(addr) : "memory")
#define POLL_ISSUE64(v, addr) asm volatile("global_load_dwordx2 %0, %1, off " POLLBITS : "+v"(v) : "v"(addr) : "memory")
#define POLL_WAIT(v, n) asm volatile("s_waitcnt vmcnt(" #n ")" : "+v"(v) : : "memory")
#define POLL_DRAIN(a, b, c) asm volatile("s_waitcnt vmcnt(0)" : "+v"(a), "+v"(b), "+v"(c) : : "memory")
template <class F>
__device__ __forceinline__ void wait_gmax(char* ws, int rep, u32 call, int lane, u64 t0, F&& publish) {
    u32 gmax = 0;
    const char* ga = ws + WS_AMAX2 + rep * 128 + min(lane, NR) * 8;
    u64 g0 = 0, g1 = 0, g2 = 0;
    int res = 0;
    VMWAIT(0);
    POLL_ISSUE64(g0, ga);
    __builtin_amdgcn_s_sleep(POLL_STAG);
    POLL_ISSUE64(g1, ga);
    __builtin_amdgcn_s_sleep(POLL_STAG);
    POLL_ISSUE64(g2, ga);
#define GCHECK(g) { POLL_WAIT(g, 2); const bool tmo = __builtin_amdgcn_readlane((u32)g, NR) == call; \
        if (__all(lane >= NR || (u32)(g >> 32) == call)) { gmax = rfl(row16_max(lane < NR ? (u32)g : 0)); res = tmo ? 2 : 1; break; } \
        if (tmo || now() - t0 > DEADLINE_TICKS) { res = 2; break; } \
        POLL_ISSUE64(g, ga); }
    while (true) { GCHECK(g0) GCHECK(g1) GCHECK(g2) }
    publish(res == 1, gmax);
    POLL_DRAIN(g0, g1, g2);
}
template <int N> __device__ __forceinline__ void vmwait() { asm volatile("s_waitcnt vmcnt(%0)" ::"n"(N) : "memory"); }
template <int U> __device__ __forceinline__ void pace() {
    vmwait<PW - 1 + (PW > U ? 5 * ((PW - U + 4) / 5) : 0)>();
}
constexpr bool tail_flag(int c) { return c == 75 || c >= 77; }
constexpr int tail_younger(int need, int next, int lastflag) {
    int n = next - 1 - need;
    for (int c = 75; c <= lastflag; ++c)
        if (tail_flag(c) && c + PW - 1 > need) ++n;
    return n;
}

__global__ void __launch_bounds__(832, 1) ag_gemm_q8_kernel(Params p) {
    __shared__ __attribute__((aligned(16))) char smem[RING_B + 128];
    volatile LAS u32* ctl = (volatile LAS u32*)(smem + RING_B);
    const int lane = threadIdx.x & 63;
    const int wave = rfl(threadIdx.x >> 6);
    const int b = blockIdx.x;
    const int rank = p.rank & 7;
    const int grp = rank >> 2, me = rank & 3;
    char* const myws = p.ws[rank];
    const u32 td = p.tile[b];

    if (td >> T_PUSH & 1) {
        if (wave >= 7) return;
        const int i = b & 7, r = b >> 3, d = wave + 1, dst = (rank ^ d) & 7;
        const bool cross = d >= 4;
        const u32 epoch = ld_sys(myws + WS_EPOCH);
        u32 so[5], dof[5];
        bool isw[5];
#pragma unroll
        for (int jj = 0; jj < 5; ++jj) {
            const int t = (4 * jj + r) / 5, u = (4 * jj + r) % 5;
            isw[jj] = cross && u >= 3;
            if (isw[jj]) {
                so[jj] = wsrc_off(dst & 3, 2 * i + u - 3, lane) + t * 65536;
                dof[jj] = WS_W + me * WSLOT_B + t * 16384 + (2 * i + u - 3) * 1024 + lane * 16;
            } else {
                so[jj] = xsrc_off((cross ? 5 : 0) + u, i, lane) + t * 128;
                dof[jj] = WS_X + ((cross ? 20 + 3 * me : 5 * me) + u) * XSLOT_B + t * 8192 + i * 1024 + lane * 16;
            }
        }
        const u32 fl = WS_DFLAG + (rank * 32 + r * 8 + i) * FSTRIDE;
        auto rx = __builtin_amdgcn_make_buffer_rsrc((void*)p.x, 0, IN_B, 0x00020000);
        auto rw = __builtin_amdgcn_make_buffer_rsrc((void*)p.w, 0, IN_B, 0x00020000);
        auto rd = __builtin_amdgcn_make_buffer_rsrc((void*)p.ws[dst], 0, WS_END, 0x00020000);
        u32x4 v[3][5];
#define PLOAD(buf, g) do { const u32 g_ = min((g), 15); _Pragma("unroll") for (int jj = 0; jj < 5; ++jj) \
        v[buf][jj] = __builtin_amdgcn_raw_buffer_load_b128(isw[jj] ? rw : rx, so[jj], g_ * (isw[jj] ? 262144 : 512), 0); CFENCE(); } while (0)
#define PST(buf, g, jj) __builtin_amdgcn_raw_buffer_store_b128(v[buf][jj], rd, dof[jj], (g) * (isw[jj] ? 65536 : 32768), 17)
#define PFLAG(c) __builtin_amdgcn_raw_buffer_store_b32(fbase + (c), rd, lane == 0 ? fl : 0xfffffff0u, 0, 17)
#define PSTEP(buf, g, jj) do { pace<jj>(); \
        if (jj == PW - 1 && (g) > 0 && (((g) & 1) == 0 || (g) >= FDENSE)) { vmwait<PW - 1>(); PFLAG((g) * 5); } \
        if (jj == PW + 2 && (g) == 14 && FDENSE < 15) { vmwait<PW - 1>(); PFLAG(73); } \
        PST(buf, g, jj); } while (0)
#define PSTORE(buf, g) do { PSTEP(buf, g, 0); PSTEP(buf, g, 1); PSTEP(buf, g, 2); PSTEP(buf, g, 3); PSTEP(buf, g, 4); CFENCE(); } while (0)
#define PTAIL(k) do { constexpr int c_ = (k) - PW + 1; \
        vmwait<tail_younger(c_ - 1, (k) < NPIECE ? (k) : NPIECE, c_ - 1)>(); \
        if (c_ >= 75 && tail_flag(c_)) PFLAG(c_); \
        if ((k) < NPIECE) PST(0, 15, (k) < NPIECE ? (k) - 75 : 0); } while (0)
        static_assert(PW >= 1 && PW <= 5, "window");
        PLOAD(0, 0); PLOAD(1, 1);
        const u32 call = rfl(epoch) + 1;
        const u32 fbase = (call - 1) * NPIECE;
        if (wave == 0) { STAMP(4); STAMP(8); }
        for (int g = 0; g < 15; g += 3) {
            PLOAD(2, g + 2); PSTORE(0, g);
            PLOAD(0, g + 3); PSTORE(1, g + 1);
            PLOAD(1, g + 4); PSTORE(2, g + 2);
        }
        PTAIL(75); PTAIL(76); PTAIL(77); PTAIL(78); PTAIL(79);
        if (PW > 1) PTAIL(80);
        if (PW > 2) PTAIL(81);
        if (PW > 3) PTAIL(82);
        if (PW > 4) PTAIL(83);
        PTAIL(79 + PW);
        STAMP(wave < 4 ? 9 + wave : 15);
        const u64 t0 = now();
        if (b == 0 && wave < NAGG) {
            auto r1 = __builtin_amdgcn_make_buffer_rsrc((void*)(myws + WS_AMAX1), 0, NWG * 8, 0x00020000);
            u64 dstp = 0;
#pragma unroll
            for (int q = 0; q < NR; ++q) if ((lane & 7) == q) dstp = (u64)p.ws[q];
            dstp += WS_AMAX2 + (lane >> 3) * 128 + rank * 8;
            asm volatile("" : "+v"(dstp));
            u32 m = 0;
            while (true) {
                const u32x4 g0 = __builtin_amdgcn_raw_buffer_load_b128(r1, lane * 16, 0, 17);
                const u32x4 g1 = __builtin_amdgcn_raw_buffer_load_b128(r1, lane * 16 + 1024, 0, 17);
                const u32 tm = ld_sys(myws + WS_AMAX2 + TMO_OFF);
                if (__all((lane < 16 || (g0.y == call && g0.w == call)) && g1.y == call && g1.w == call)) {
                    m = max(lane < 16 ? 0u : max(g0.x, g0.z), max(g1.x, g1.z));
                    break;
                }
                if (rfl(tm) == call || now() - t0 > DEADLINE_TICKS) { raise_tmo(myws, call, lane, 3); return; }
            }
            m = wave_max(m);
#pragma unroll
            for (int q = 0; q < NREP / 8; ++q)
                __hip_atomic_store((GAS u64*)(dstp + q * 1024), ((u64)call << 32) | m, __ATOMIC_RELAXED, __HIP_MEMORY_SCOPE_SYSTEM);
            STAMP(7);
        } else if (wave == 6) {
            bool ok;
            wait_gmax(myws, b % NREP, call, lane, t0, [&](bool good, u32) { ok = good; });
            if (b == 0 && lane == 0) st_agent(myws + WS_EPOCH, call);
            const bool pf = lane < 4 && !(b >= 20 + 3 * me && b < 23 + 3 * me);
            auto rf = __builtin_amdgcn_make_buffer_rsrc((void*)myws, 0, WS_X, 0x00020000);
            const u32 fo = pf ? WS_OFLAG + (lane * 32 + b) * 128 : WS_AMAX2 + (b % NREP) * 128 + TMO_OFF;
            while (ok) {
                const u32x4 f = __builtin_amdgcn_raw_buffer_load_b128(rf, fo, 0, 17);
                if (__all(!pf || ((int)(f.x - call) >= 0 && (int)(f.y - call) >= 0 && (int)(f.z - call) >= 0 && (int)(f.w - call) >= 0))) break;
                if (__builtin_amdgcn_readlane(f.x, 4) == call || now() - t0 > DEADLINE_TICKS) ok = false;
            }
            if (!ok) {
                raise_tmo(myws, call, lane, 4);
                if (lane == 0) __hip_atomic_store((GAS char*)p.out[rank] + (size_t)rank * 512 * 512, (char)-128, __ATOMIC_RELAXED, __HIP_MEMORY_SCOPE_SYSTEM);
            }
            STAMP(3);
        }
        return;
    }

    const int srank = td >> T_SRANK & 7, panel = td >> T_PANEL & 7, slot = td >> T_SLOT & 31, ci = td >> T_CI & 3;
    const bool wlocal = td >> T_WLOCAL & 1, dbl = td >> T_DBL & 1, xlocal = srank == rank;
    const int panelb = td >> T_PANELB & 7;
    const int orank = wlocal ? rank : 4 * (grp ^ 1) + ci;
    const int nstage = dbl ? 4 : 6;
    const u32 stage_b = dbl ? 32768 : 24576;

    if (wave == 12) {
        const u32 call = ld_sys(myws + WS_EPOCH) + 1;
        if (lane < 32) ctl[lane] = lane == C_CALL ? call : (lane >= 4 && lane < 8 && !dbl) ? 4096 : ((lane == C_AVX && xlocal) || (lane == C_AVW && wlocal)) ? NSTEP : 0;
        asm volatile("s_waitcnt lgkmcnt(0)" ::: "memory");
        __builtin_amdgcn_s_barrier();
        STAMP(4);
        const u64 t0 = now();
        const int rep = b % NREP;
        const u32 fbase = (call - 1) * NPIECE;
        const int l32 = lane & 31;
        const int frank = lane < 32 ? srank : orank;
        const bool pf = frank != rank;
        const char* tmoa = myws + WS_AMAX2 + rep * 128 + TMO_OFF;
        const char* fa = pf ? myws + WS_DFLAG + (frank * 32 + l32) * FSTRIDE : tmoa;
        bool bad = false;
        int got = 0, sum = 0;
        u32 npoll = 0;
#define DCHECK(f) { ++npoll; const u32 a_ = row16_max(~(pf ? max((int)(f - fbase), 0) * 4 + (l32 >> 3) : 5 * NSTEP)); \
            const int ax_ = min(NSTEP, (int)(~max((u32)__builtin_amdgcn_readlane(a_, 0), (u32)__builtin_amdgcn_readlane(a_, 16)) / 5)); \
            const int aw_ = min(NSTEP, (int)(~max((u32)__builtin_amdgcn_readlane(a_, 32), (u32)__builtin_amdgcn_readlane(a_, 48)) / 5)); \
            if (ax_ + aw_ > sum) { sum = ax_ + aw_; got = min(ax_, aw_); if (lane == 0) *(volatile LAS u32x2*)&ctl[C_AVX] = (u32x2){(u32)ax_, (u32)aw_}; if (got == NSTEP) break; } \
            else if (now() - t0 > 20000   && (rfl(ld_sys(tmoa)) == call || now() - t0 > DEADLINE_TICKS)) { bad = true; break; } }
        while (got < 48) {
            u32 f = 0;
            POLL_ISSUE32(f, fa);
            POLL_WAIT(f, 0);
            DCHECK(f)
            __builtin_amdgcn_s_sleep(POLL_SLEEP);
        }
        if (!bad && got < NSTEP) {
            u32 f0 = 0, f1 = 0, f2 = 0;
            POLL_ISSUE32(f0, fa);
            __builtin_amdgcn_s_sleep(POLL_STAG);
            POLL_ISSUE32(f1, fa);
            __builtin_amdgcn_s_sleep(POLL_STAG);
            POLL_ISSUE32(f2, fa);
            while (true) {
                POLL_WAIT(f0, 2); DCHECK(f0) POLL_ISSUE32(f0, fa);
                POLL_WAIT(f1, 2); DCHECK(f1) POLL_ISSUE32(f1, fa);
                POLL_WAIT(f2, 2); DCHECK(f2) POLL_ISSUE32(f2, fa);
            }
            POLL_DRAIN(f0, f1, f2);
        }
        STAMP(5);
        STAMPV(14, npoll);
        auto publish = [&](bool good, u32 gmax) {
            if (lane == 0) { ctl[C_GMAX] = gmax; CFENCE(); ctl[C_GRDY] = good ? 1 : 2; if (!good) { ctl[C_AVX] = NSTEP; ctl[C_AVW] = NSTEP; } }
            if (!good) raise_tmo(myws, call, lane, bad ? 0x100 + got + (sum << 16) : 2);
        };
        if (!bad) wait_gmax(myws, rep, call, lane, t0, publish);
        else publish(false, 0);
        STAMP(6);
        return;
    }

    const u32 lds0 = (u32)(uintptr_t)(LAS char*)smem;
    if (wave >= 8) {
        const u32x4 srdx = xlocal ? make_srd(p.x, IN_B) : make_srd(myws + WS_X + slot * XSLOT_B, XSLOT_B);
        const u32x4 srdw = wlocal ? make_srd(p.w, IN_B) : make_srd(myws + WS_W + ci * WSLOT_B, WSLOT_B);
        const u32x4 srdb = make_srd(p.x, IN_B);
        const u32 sxs = xlocal ? 128 : 8192, sws = wlocal ? 65536 : 16384;
        const u32 gxs = xlocal ? 32 * KDIM * 2 : 4096, gws = wlocal ? 16384 : 4096;
        u32 vx[4], vw[4], vb[4];
#pragma unroll
        for (int i = 0; i < 4; ++i) {
            vx[i] = xlocal ? xsrc_off(panel, i, lane) - i * 1024 : lane * 16;
            vw[i] = wlocal ? wsrc_off(ci, i, lane) - i * 1024 : lane * 16;
            vb[i] = xsrc_off(panelb, i, lane) - i * 1024;
        }
        __builtin_amdgcn_s_barrier();
        int sl = wave - 8;
        for (int t = wave - 8; t < NSTEP; t += 4) {
            while (true) {
                const u32x4 d0 = *(volatile LAS u32x4*)&ctl[C_DONE], d1 = *(volatile LAS u32x4*)&ctl[C_DONE + 4];
                const int dmin = min(min(min(d0.x, d0.y), min(d0.z, d0.w)), min(min(d1.x, d1.y), min(d1.z, d1.w)));
                if (rfl(dmin + nstage > t)) break;
                __builtin_amdgcn_s_sleep(1);
            }
            CFENCE();
            const u32 st = lds0 + (u32)sl * stage_b;
            if (dbl) {
                dma4(st + 24576, vb[0], vb[1], vb[2], vb[3], srdb, t * 128);
                dma4(st + 28672, vb[0], vb[1], vb[2], vb[3], srdb, t * 128 + 32 * KDIM * 2);
            }
            bool needx = true, needw = true;
            while (true) {
                const u32x2 av = *(volatile LAS u32x2*)&ctl[C_AVX];
                CFENCE();
                if (needw && (int)rfl(av.y) > t) {
                    dma4(st + 8192, vw[0], vw[1], vw[2], vw[3], srdw, t * sws);
                    dma4(st + 12288, vw[0], vw[1], vw[2], vw[3], srdw, t * sws + gws);
                    dma4(st + 16384, vw[0], vw[1], vw[2], vw[3], srdw, t * sws + 2 * gws);
                    dma4(st + 20480, vw[0], vw[1], vw[2], vw[3], srdw, t * sws + 3 * gws);
                    needw = false;
                }
                if (needx && (int)rfl(av.x) > t) {
                    dma4(st, vx[0], vx[1], vx[2], vx[3], srdx, t * sxs);
                    dma4(st + 4096, vx[0], vx[1], vx[2], vx[3], srdx, t * sxs + gxs);
                    needx = false;
                }
                if (!needx && !needw) break;
                __builtin_amdgcn_s_sleep(1);
            }
            VMWAIT(0);
            if (lane == 0) ctl[C_READY + (wave - 8)] = t + 1;
            sl += 4;
            if (sl >= nstage) sl -= nstage;
        }
        return;
    }

    const int tb = wave >> 2, wm = (wave >> 1) & 1, wn = wave & 1;
    const int l15 = lane & 15, g4 = lane >> 4;
    u32 ax[2], aw[4];
#pragma unroll
    for (int kk = 0; kk < 2; ++kk) ax[kk] = tb * 24576 + (32 * wm + l15) * 128 + (((4 * kk + g4) ^ (l15 >> 1)) << 4);
    {
        const int q = (lane >> 2) & 3, pp = lane & 3;
#pragma unroll
        for (int nt = 0; nt < 4; ++nt)
            aw[nt] = 8192 + (8 * g4 + q) * 256 + (((8 * wn + 4 * (pp >> 1) + nt) ^ (q | ((g4 & 1) << 3))) << 4) + 8 * (pp & 1);
    }
    u64 o_ws = (u64)p.ws[orank & 7], o_out = (u64)p.out[orank & 7];
    asm volatile("" : "+s"(o_ws), "+s"(o_out));
    f32x4 acc[2][4];
#pragma unroll
    for (int mt = 0; mt < 2; ++mt)
#pragma unroll
        for (int nt = 0; nt < 4; ++nt) acc[mt][nt] = (f32x4){0.f, 0.f, 0.f, 0.f};

    __builtin_amdgcn_s_barrier();
    if (tb && !dbl) return;
    if (wave == 0) STAMP(0);
    const u32 call = rfl(ctl[C_CALL]);
    u32 so = 0;
    for (int s = 0; s < NSTEP; ++s) {
        while ((int)rfl(ctl[C_READY + (s & 3)]) <= s) __builtin_amdgcn_s_sleep(1);
        CFENCE();
        LAS char* st = (LAS char*)smem + so;
        s16x8 xf[2][2], wf[2][4];
#pragma unroll
        for (int kk = 0; kk < 2; ++kk) {
#pragma unroll
            for (int mt = 0; mt < 2; ++mt) xf[kk][mt] = *(LAS s16x8*)(st + ax[kk] + mt * 2048);
#pragma unroll
            for (int nt = 0; nt < 4; ++nt) {
                const s16x4 lo = __builtin_amdgcn_ds_read_tr16_b64_v4i16((LAS s16x4*)(st + aw[nt] + kk * 8192));
                const s16x4 hi = __builtin_amdgcn_ds_read_tr16_b64_v4i16((LAS s16x4*)(st + aw[nt] + kk * 8192 + 1024));
                wf[kk][nt] = __builtin_shufflevector(lo, hi, 0, 1, 2, 3, 4, 5, 6, 7);
            }
        }
        CFENCE();
        if (lane == 0) ctl[C_DONE + wave] = s + 1;
        CFENCE();
#pragma unroll
        for (int kk = 0; kk < 2; ++kk)
#pragma unroll
            for (int mt = 0; mt < 2; ++mt)
#pragma unroll
                for (int nt = 0; nt < 4; ++nt)
                    acc[mt][nt] = __builtin_amdgcn_mfma_f32_16x16x32_bf16(wf[kk][nt], xf[kk][mt], acc[mt][nt], 0, 0, 0);
        so += stage_b;
        if (so == nstage * stage_b) so = 0;
    }
    if (wave == 0) STAMP(1);
    {
        float m = 0.f;
#pragma unroll
        for (int mt = 0; mt < 2; ++mt)
#pragma unroll
            for (int nt = 0; nt < 4; ++nt)
#pragma unroll
                for (int r = 0; r < 4; ++r) m = fmaxf(m, fabsf(acc[mt][nt][r]));
        const u32 mu = wave_max(__float_as_uint(m));
        if (lane == 0) {
            __hip_atomic_fetch_max((LAS u32*)&ctl[C_WMAX], mu, __ATOMIC_RELAXED, __HIP_MEMORY_SCOPE_WORKGROUP);
            CFENCE();
            const u32 old = __hip_atomic_fetch_add((LAS u32*)&ctl[C_WCNT], 1u, __ATOMIC_RELAXED, __HIP_MEMORY_SCOPE_WORKGROUP);
            CFENCE();
            const u32 wmax = ctl[C_WMAX];
            if (old == (dbl ? 7u : 3u)) __hip_atomic_store((GAS u64*)(myws + WS_AMAX1 + b * 8), ((u64)call << 32) | wmax, __ATOMIC_RELAXED, __HIP_MEMORY_SCOPE_AGENT);
        }
    }
    u32 rdy;
    while ((rdy = rfl(ctl[C_GRDY])) == 0) __builtin_amdgcn_s_sleep(1);
    CFENCE();
    if (wave == 0) STAMP(2);
    const float gmax = __uint_as_float(ctl[C_GMAX]);
    const float inv = gmax > 0.f ? 127.0f / gmax : 0.f;
    auto ro = __builtin_amdgcn_make_buffer_rsrc((void*)o_out, 0, OUT_B, 0x00020000);
    const int rowblk = tb ? rank * 8 + panelb : srank * 8 + panel;
    const u32 obase = (u32)(rowblk * 64 + 32 * wm + l15) * 512u + (wlocal ? ci : me) * 128 + wn * 64 + g4 * 16;
#pragma unroll
    for (int mt = 0; mt < 2; ++mt) {
        u32 dw[4];
#pragma unroll
        for (int nt = 0; nt < 4; ++nt) {
            u32 pk = 0;
#pragma unroll
            for (int r = 0; r < 4; ++r) {
                const int q = (int)__builtin_rintf(fminf(fmaxf(acc[mt][nt][r] * inv, -127.f), 127.f));
                pk |= ((u32)q & 0xffu) << (8 * r);
            }
            dw[nt] = pk;
        }
        const auto a = __builtin_amdgcn_permlane16_swap(dw[0], dw[2], false, false);
        const auto c = __builtin_amdgcn_permlane16_swap(dw[1], dw[3], false, false);
        u32x4 o = {a[0], a[1], c[0], c[1]};
        if (rdy != 1) o = (u32x4){0x80808080u, 0x80808080u, 0x80808080u, 0x80808080u};
        __builtin_amdgcn_raw_buffer_store_b128(o, ro, obase + mt * 16 * 512, 0, 17);
    }
    if (!wlocal) {
        VMWAIT(0);
        if (lane == 0) st_sys((char*)o_ws + WS_OFLAG + (me * 32 + slot) * 128 + wave * 4, call);
    }
    if (wave == 0) STAMP(13);
}

static void build_tiles(int rank, u32* tile) {
    const int grp = rank >> 2;
    for (int b = 0; b < 32; ++b) tile[b] = 1u << T_PUSH;
    for (int h = 0; h < 2; ++h) {
        int rows[4][11], nrows[4] = {0, 0, 0, 0}, nn = 0;
        for (int slot = 0; slot < 32; ++slot) {
            const int srank = slot < 20 ? 4 * grp + slot / 5 : 4 * (grp ^ 1) + (slot - 20) / 3;
            const bool special = srank == rank || (slot >= 20 && ((slot - 20) / 3) >> 1 == h);
            const int q = special ? 3 : nn++ / 7;
            rows[q][nrows[q]++] = slot;
        }
        for (int q = 0; q < 4; ++q) {
            int n = 0, nb[2] = {0, 0};
            for (int k = 0; k < nrows[q]; ++k) {
                const int slot = rows[q][k];
                const int srank = slot < 20 ? 4 * grp + slot / 5 : 4 * (grp ^ 1) + (slot - 20) / 3;
                const int panel = slot < 20 ? slot % 5 : 5 + (slot - 20) % 3;
                const u32 base = srank << T_SRANK | panel << T_PANEL | slot << T_SLOT;
                for (int c = 0; c < 2; ++c) {
                    const int ci = 2 * h + c;
                    if (srank != rank) {
                        u32 t = base | 1u << T_WLOCAL | ci << T_CI;
                        if (nb[c] < 2) t |= 1u << T_DBL | (2 * q + nb[c]++) << T_PANELB;
                        tile[(4 + n++) * 8 + 2 * q + h] = t;
                    }
                    if (srank != 4 * (grp ^ 1) + ci) tile[(4 + n++) * 8 + 2 * q + h] = base | ci << T_CI;
                }
            }
        }
    }
}

extern "C" void kernel_launch(int rank, int nranks, ncclComm_t comm,
                              hipStream_t stream,
                              void* const* d_in, const int* in_sizes, int n_in,
                              void* d_out, int out_size,
                              const int* in_dtypes, int out_dtype,
                              void* const* d_out_all,
                              void* const* d_ws_all, size_t ws_bytes) {
    if (nranks != NR || (unsigned)rank >= (unsigned)NR || n_in != 2 || ws_bytes < WS_END ||
        in_sizes[0] != (int)(IN_B / 2) || in_sizes[1] != (int)(IN_B / 2) || out_size != (int)OUT_B) {
        fprintf(stderr, "kernel_launch: unexpected problem shape\n");
        return;
    }
    Params p;
    p.x = (const char*)d_in[0];
    p.w = (const char*)d_in[1];
    for (int i = 0; i < NR; ++i) { p.ws[i] = (char*)d_ws_all[i]; p.out[i] = (char*)d_out_all[i]; }
    p.rank = rank;
    build_tiles(rank, p.tile);
    ag_gemm_q8_kernel<<<NWG, 832, 0, stream>>>(p);
}
```

```cpp
#include <hip/hip_runtime.h>
#include <rccl/rccl.h>
#include <cstdio>
#include <cstdint>

typedef unsigned int u32;
typedef unsigned long long u64;
typedef __attribute__((ext_vector_type(4))) unsigned int u32x4;
typedef __attribute__((ext_vector_type(2))) unsigned int u32x2;
typedef __attribute__((ext_vector_type(4))) float f32x4;
typedef __attribute__((ext_vector_type(8))) short s16x8;
typedef __attribute__((ext_vector_type(4))) short s16x4;
#define LAS __attribute__((address_space(3)))
#define GAS __attribute__((address_space(1)))

constexpr int TAU16 = 897;
constexpr int TAUMIN = 50;
constexpr int LATE_TICKS = 40;
constexpr int TAULATE = 45;
constexpr int TSTART = 70;
constexpr int TPEND = 4600;
constexpr int CDELAY = 400;
constexpr int SPN = 8;
constexpr int PROBE_AT = 52;
constexpr int GDELAY = 60;
constexpr int NEARLY = 9;
constexpr int ETAU = 28;
constexpr int OTAU = 28;
constexpr int WLAT = 70;
#define FLAGS 12, 24, 34, 42, 48, 53, 56
#define POLL_SLEEP 4
#define RETRY_SLEEP 4

constexpr int NR = 8;
constexpr int KDIM = 4096;
constexpr int NSTEP = 64;
constexpr int RING_B = 6 * 24576;
constexpr int SP0 = NSTEP - SPN;
constexpr int NWG = 256;
constexpr int NTHREADS = 11 * 64;
constexpr int NPIECE = 80;
constexpr u64 DEADLINE_TICKS = 100000000ull;
template <int... S> constexpr u64 flag_mask() { return ((1ull << (S - 1)) | ... | (1ull << 63)); }
constexpr u64 FLAG_AT = flag_mask<FLAGS>();

constexpr u32 WS_EPOCH = 0;
constexpr u32 WS_TMO = 128;
constexpr u32 WS_AMAX2 = 8192;
constexpr u32 WS_EFLAG = 32768;
constexpr u32 WS_DFLAG = 65536;
constexpr u32 WS_AMAX1 = 131072;
constexpr u32 WS_X = 1u << 20;
constexpr u32 XSLOT_B = (NSTEP + SPN) * 8192;
constexpr u32 WS_W = WS_X + 32 * XSLOT_B;
constexpr u32 WSLOT_B = (NSTEP + SPN) * 16384;
constexpr u32 WS_E16 = WS_W + 4 * WSLOT_B;
constexpr u32 WS_END = WS_E16 + (1u << 20);
constexpr u32 IN_B = 4u << 20;
constexpr u32 OUT_B = 2u << 20;

constexpr int C_DONE = 0, C_READY = 8, C_AVX = 12, C_AVW = 13, C_SPX = 14, C_SPW = 15, C_INV = 16, C_GRDY = 17, C_CALL = 18;

struct Params {
    const char* x;
    const char* w;
    char* ws[NR];
    char* out[NR];
    int rank;
    int pad;
    u32 push[7][4][5][2];
    u32 tile[NWG];
    u32 inpos[NWG];
};
constexpr u32 T_SRANK = 0, T_PANEL = 3, T_SLOT = 6, T_WLOCAL = 11, T_CI = 12, T_DBL = 14, T_PANELB = 15, T_INA = 18, T_INB = 25;

__device__ __forceinline__ u32 rfl(u32 v) { return __builtin_amdgcn_readfirstlane(v); }
__device__ __forceinline__ u32x4 make_srd(const void* base, u32 bytes) {
    const u64 a = (u64)base;
    return (u32x4){rfl((u32)a), rfl((u32)(a >> 32) & 0xffffu), bytes, 0x00020000u};
}
__device__ __forceinline__ u32 ld_sys(const void* p) { return __hip_atomic_load((GAS const u32*)p, __ATOMIC_RELAXED, __HIP_MEMORY_SCOPE_SYSTEM); }
__device__ __forceinline__ u64 ld_sys64(const void* p) { return __hip_atomic_load((GAS const u64*)p, __ATOMIC_RELAXED, __HIP_MEMORY_SCOPE_SYSTEM); }
__device__ __forceinline__ void st_sys(void* p, u32 v) { __hip_atomic_store((GAS u32*)p, v, __ATOMIC_RELAXED, __HIP_MEMORY_SCOPE_SYSTEM); }
__device__ __forceinline__ void st_sys64(void* p, u64 v) { __hip_atomic_store((GAS u64*)p, v, __ATOMIC_RELAXED, __HIP_MEMORY_SCOPE_SYSTEM); }
__device__ __forceinline__ u64 now() { return __builtin_amdgcn_s_memrealtime(); }
template <int N> __device__ __forceinline__ void vmwait() { asm volatile("s_waitcnt vmcnt(%0)" ::"n"(N) : "memory"); }
#define CFENCE() asm volatile("" ::: "memory")
#define SCHED() __builtin_amdgcn_sched_barrier(0)


template <int CTRL> __device__ __forceinline__ u32 dpp(u32 v) { return __builtin_amdgcn_update_dpp(0u, v, CTRL, 0xf, 0xf, true); }
__device__ __forceinline__ u32 row16_max(u32 v) {
    v = max(v, dpp<0xB1>(v)); v = max(v, dpp<0x4E>(v)); v = max(v, dpp<0x141>(v)); v = max(v, dpp<0x140>(v)); return v;
}
__device__ __forceinline__ u32 wave_max(u32 v) {
    v = row16_max(v);
    const auto a = __builtin_amdgcn_permlane16_swap(v, v, false, false);
    v = max(a[0], a[1]);
    const auto c = __builtin_amdgcn_permlane32_swap(v, v, false, false);
    return max(c[0], c[1]);
}

__device__ __forceinline__ u32 xsrc_off(int panel, int piece, int lane) {
    const int r = piece * 8 + (lane >> 3);
    return (u32)(panel * 64 + r) * (KDIM * 2) + (u32)(((lane & 7) ^ ((r >> 1) & 7)) << 4);
}
__device__ __forceinline__ u32 wsrc_off(int blk, int piece, int lane) {
    const int k = piece * 4 + (lane >> 4);
    return (u32)k * 1024u + (u32)blk * 256u + (u32)(((lane & 15) ^ ((k & 3) | (((k >> 3) & 1) << 3))) << 4);
}

__device__ __forceinline__ void dma4(u32 lds, u32 v0, u32 v1, u32 v2, u32 v3, u32x4 srd, u32 soff) {
    u32 keep;
    asm volatile(
        "s_mov_b32 %0, m0\n\t"
        "s_mov_b32 m0, %1\n\t"
        "s_nop 0\n\t"
        "buffer_load_dwordx4 %2, %6, %7 offen sc0 sc1 lds\n\t"
        "buffer_load_dwordx4 %3, %6, %7 offen offset:1024 sc0 sc1 lds\n\t"
        "buffer_load_dwordx4 %4, %6, %7 offen offset:2048 sc0 sc1 lds\n\t"
        "buffer_load_dwordx4 %5, %6, %7 offen offset:3072 sc0 sc1 lds\n\t"
        "s_mov_b32 m0, %0"
        : "=&s"(keep)
        : "s"(lds), "v"(v0), "v"(v1), "v"(v2), "v"(v3), "s"(srd), "s"(soff)
        : "memory");
}

__device__ __forceinline__ void raise_tmo(char* ws, u32 call, int lane) {
    if (lane == 0) __hip_atomic_store((GAS u32*)(ws + WS_TMO), call, __ATOMIC_RELAXED, __HIP_MEMORY_SCOPE_AGENT);
}
__device__ __forceinline__ bool wait_gmax(char* ws, u32 call, int lane, u64 t0, u32& gmax) {
    const char* ga = ws + WS_AMAX2 + lane * 128;
    bool have = false;
    u32 val = 0;
    while (true) {
        const u64 g = ld_sys64(have ? ws + WS_TMO : ga);
        const bool tmo = __any(have) ? have && (u32)g == call : ld_sys(ws + WS_TMO) == call;
        if (!have && (u32)(g >> 32) == call) { have = true; val = (u32)g; }
        if (__all(have)) { gmax = wave_max(val); return !__any(tmo); }
        if (__any(tmo) || now() - t0 > DEADLINE_TICKS) return false;
    }
}

__global__ void __launch_bounds__(NTHREADS, 1) ag_gemm_q8_kernel(Params p) {
    __shared__ __attribute__((aligned(16))) char smem[RING_B + 128];
    volatile LAS u32* ctl = (volatile LAS u32*)(smem + RING_B);
    const u64 t_entry = now();
    const int lane = threadIdx.x & 63;
    const int wave = rfl(threadIdx.x >> 6);
    const int b = blockIdx.x;

    if (b < 32) {
        const int i = b & 7, r = b >> 3;
        if (wave < 7) {
            __builtin_amdgcn_s_setprio(3);
            const u32 xl = xsrc_off(0, i, lane), wl = wsrc_off(0, 2 * i, lane);
            u32 so[5], dof[5];
            bool isw[5];
#pragma unroll
            for (int jj = 0; jj < 5; ++jj) {
                so[jj] = p.push[wave][r][jj][0];
                dof[jj] = p.push[wave][r][jj][1];
                isw[jj] = so[jj] >> 31;
                so[jj] &= 0x7fffffffu;
            }
            auto rx = __builtin_amdgcn_make_buffer_rsrc((void*)p.x, 0, IN_B, 0x00020000);
            auto rw = __builtin_amdgcn_make_buffer_rsrc((void*)p.w, 0, IN_B, 0x00020000);
            auto rd = __builtin_amdgcn_make_buffer_rsrc((void*)p.ws[wave + 1], 0, WS_END, 0x00020000);
            u32x4 v[2][5];
#define PLD(buf, g, jj) v[buf][jj] = __builtin_amdgcn_raw_buffer_load_b128(isw[jj] ? rw : rx, isw[jj] ? wl : xl, so[jj] + (g) * (isw[jj] ? 262144 : 512), 0)
#define PST(buf, g, jj) __builtin_amdgcn_raw_buffer_store_b128(v[buf][jj], rd, (isw[jj] ? i * 2048 : i * 1024) + lane * 16, dof[jj] + (g) * (isw[jj] ? 65536 : 32768), 17)
#define PROG(c) do { if (lane == 0) ctl[wave] = (c) * 4 + r; } while (0)
#define TPACE(k) do { const u32 sch_ = tbase + (((k) * TAU16) >> 4); u32 t_; \
            while (t_ = (u32)now(), (int)(t_ - sch_) < 0 || (int)(t_ - tlast) < (late ? TAULATE : TAUMIN)) __builtin_amdgcn_s_sleep(1); \
            tlast = t_; late = (int)(t_ - sch_) > LATE_TICKS; } while (0)
#define PSTEP(buf, g, jj) do { TPACE((g) * 5 + jj); \
            if (late) { vmwait<4>(); PROG(max((g) * 5 + jj - 4, 0)); } else { vmwait<2>(); PROG(max((g) * 5 + jj - 2, 0)); } \
            PST(buf, g, jj); PLD(buf, (g) + 2, jj); CFENCE(); } while (0)
#define PCYCLE(buf, g) do { PSTEP(buf, g, 0); PSTEP(buf, g, 1); PSTEP(buf, g, 2); PSTEP(buf, g, 3); PSTEP(buf, g, 4); } while (0)
            PLD(0, 0, 0); PLD(0, 0, 1); PLD(0, 0, 2); PLD(0, 0, 3);
            CFENCE();
            vmwait<3>();
            const u32 tbase = (u32)t_entry + TSTART + (((r * 8 + i) * TAU16) >> 9);
            u32 tlast = tbase - 1000;
            bool late = false;
            TPACE(0); PST(0, 0, 0); PLD(0, 0, 4); PLD(1, 1, 0); CFENCE();
            TPACE(1); PST(0, 0, 1); PLD(1, 1, 1); PLD(1, 1, 2); CFENCE();
            __builtin_amdgcn_s_barrier();
            TPACE(2); vmwait<5>(); PST(0, 0, 2); PLD(1, 1, 3); PLD(1, 1, 4); CFENCE();
            TPACE(3); vmwait<5>(); PST(0, 0, 3); PLD(0, 2, 0); PLD(0, 2, 1); CFENCE();
            TPACE(4); vmwait<5>(); PST(0, 0, 4); PLD(0, 2, 2); PLD(0, 2, 3); CFENCE();
            TPACE(5); vmwait<5>(); PST(1, 1, 0); PLD(0, 2, 4); PLD(1, 3, 0); CFENCE();
            TPACE(6); vmwait<5>(); PST(1, 1, 1); PLD(1, 3, 1); CFENCE();
            TPACE(7); vmwait<4>(); PST(1, 1, 2); PLD(1, 3, 2); CFENCE();
            PSTEP(1, 1, 3); PSTEP(1, 1, 4);
#pragma unroll 1
            for (int g = 2; g < 12; g += 2) {
                PCYCLE(0, g); PCYCLE(1, g + 1);
            }
            PCYCLE(0, 12);
            const int gp = (rfl(ctl[7]) & 1) * (SPN / 4);
            const int g13 = 13 + (SPN > 8 ? gp : 0);
#pragma unroll
            for (int jj = 0; jj < 5; ++jj) {
                TPACE(65 + jj);
                if (late) { vmwait<4>(); PROG(61 + jj); } else { vmwait<2>(); PROG(63 + jj); }
                PST(1, g13, jj); PLD(1, 15, jj); CFENCE();
            }
            const int g14 = 14 + (SPN > 4 ? gp : 0), g15 = 15 + gp;
#define PTAIL(buf, g, k) do { vmwait<1>(); PROG((k) - 1); TPACE(k); PST(buf, g, (k) % 5); } while (0)
            TPACE(70); vmwait<2>(); PROG(68); PST(0, g14, 0);
            PTAIL(0, g14, 71); PTAIL(0, g14, 72); PTAIL(0, g14, 73); PTAIL(0, g14, 74);
            PTAIL(1, g15, 75); PTAIL(1, g15, 76); PTAIL(1, g15, 77); PTAIL(1, g15, 78); PTAIL(1, g15, 79);
            vmwait<1>(); PROG(79);
            vmwait<0>();
            PROG(NPIECE);
            return;
        }
        if (wave > 7) return;
        const int rank = p.rank & 7;
        char* const myws = p.ws[0];
        const u32 call = rfl(ld_sys(myws + WS_EPOCH)) + 1;
        if (lane < 8) ctl[lane] = lane == 7 ? call : 0;
        asm volatile("s_waitcnt lgkmcnt(0)" ::: "memory");
        __builtin_amdgcn_s_barrier();
        u64 fp = 0;
#pragma unroll
        for (int q = 0; q < NR; ++q) if (((lane + 1) & 7) == q) fp = (u64)p.ws[q];
        u64 ap = fp + WS_AMAX2 + (rank * 8 + i) * 128;
        fp += WS_DFLAG + (rank * 32 + b) * 128;
        asm volatile("" : "+v"(fp), "+v"(ap));
        const u32 fbase = (call - 1) * NSTEP;
        int nxt = lane < 7 ? 1 + __builtin_ctzll(FLAG_AT) : NSTEP + 1;
        while (__any(nxt <= NSTEP)) {
            const int sd = ctl[lane & 7] / 5;
            if (lane < 7 && sd >= nxt) {
                st_sys((char*)fp, fbase + sd);
                nxt = sd < NSTEP ? sd + 1 + __builtin_ctzll(FLAG_AT >> sd) : NSTEP + 1;
            }
            if (rfl(sd) < 56) __builtin_amdgcn_s_sleep(1);
        }
        if (r == 0) {
            const u64 t0 = now();
            const char* ga = myws + WS_AMAX1 + (i * 128 + lane) * 128;
            bool h0 = false, h1 = false;
            u32 val = 0;
            while (true) {
                const u64 g0 = ld_sys64(h0 ? myws + WS_TMO : ga), g1 = ld_sys64(h1 ? myws + WS_TMO : ga + 8192);
                const bool tmo = __any(h0) ? h0 && (u32)g0 == call : ld_sys(myws + WS_TMO) == call;
                if (!h0 && (u32)(g0 >> 32) == call) { h0 = true; val = max(val, (u32)g0); }
                if (!h1 && (u32)(g1 >> 32) == call) { h1 = true; val = max(val, (u32)g1); }
                if (__all(h0 && h1)) {
                    const u32 m = wave_max(val);
                    if (lane < NR) st_sys64((char*)ap, ((u64)call << 32) | m);
                    break;
                }
                if (__any(tmo) || now() - t0 > DEADLINE_TICKS) { raise_tmo(myws, call, lane); break; }
            }
            if (b == 0) {
                u32 gmax;
                wait_gmax(myws, call, lane, t0, gmax);
                if (lane == 0) __hip_atomic_store((GAS u32*)(myws + WS_EPOCH), call, __ATOMIC_RELAXED, __HIP_MEMORY_SCOPE_AGENT);
            }
        }
        return;
    }

    const int rank = p.rank & 7;
    const int grp = rank >> 2, me = rank & 3;
    char* const myws = p.ws[0];
    const u32 td = p.tile[b];
    const int srank = td >> T_SRANK & 7, panel = td >> T_PANEL & 7, slot = td >> T_SLOT & 31, ci = td >> T_CI & 3;
    const bool wlocal = td >> T_WLOCAL & 1, dbl = td >> T_DBL & 1, xlocal = srank == rank;
    const int panelb = td >> T_PANELB & 7;
    const int orank = wlocal ? rank : 4 * (grp ^ 1) + ci;
    const int opos = td >> T_INB & 31;
    const bool early = !wlocal && opos < NEARLY;
    const u32 td2 = p.inpos[b];
    const int hb = td2 >> 16 & 1;
    const int nstage = dbl ? 5 : 6;
    const u32 stage_b = dbl ? 28672 : 24576;

    if (wave == 10) {
        __builtin_amdgcn_s_setprio(3);
        const u32 call = ld_sys(myws + WS_EPOCH) + 1;
        if (lane < 32) ctl[lane] = lane == C_CALL ? call : (lane >= (dbl ? 6 : 4) && lane < 8) ? 4096 : ((lane == C_AVX && xlocal) || (lane == C_AVW && wlocal)) ? NSTEP : 0;
        asm volatile("s_waitcnt lgkmcnt(0)" ::: "memory");
        __builtin_amdgcn_s_barrier();
        const u64 t0 = now();
        const u32 fbase = (call - 1) * NSTEP;
        const int frank = lane < 32 ? srank : orank;
        const bool pf = frank != rank;
        const char* fa = myws + (pf ? WS_DFLAG + (frank * 32 + (lane & 31)) * 128 : WS_TMO);
        bool ok = true;
        int got = 0, sum = 0;
        u32 npoll = 0;
        const int pst = lane < 16 ? lane >> 3 : (lane - 16) >> 4;
        const bool pp = lane < 16 ? !xlocal : (lane < 48 && !wlocal);
        const char* pa = myws + (lane < 16 ? WS_X + slot * XSLOT_B + (SP0 + (call & 1) * SPN + pst) * 8192 + (lane & 7) * 1024
                                           : WS_W + ci * WSLOT_B + (SP0 + (call & 1) * SPN + pst) * 16384 + (lane & 15) * 1024) + 1020;
        int spx = SP0, spw = SP0;
        while (got < NSTEP) {
            const bool probe = got >= PROBE_AT && min(spx, spw) < NSTEP;
            const u32 f = !probe || got < SP0 || (npoll & 7) == 0 ? ld_sys(fa) : fbase;
            if (probe) {
                const int sb = min(min(spx, spw), NSTEP - 2);
                const u32 pv = ld_sys(pp ? pa + (sb - SP0) * (lane < 16 ? 8192 : 16384) : myws + WS_TMO);
                const u64 z = __ballot(pp && pv == 0);
                const int nx = sb + ((z & 0xffu) ? 0 : (z & 0xff00u) ? 1 : 2);
                const int nw = sb + ((z & 0xffff0000u) ? 0 : (z & 0xffff00000000ull) ? 1 : 2);
                if (nx > spx || nw > spw) {
                    spx = max(spx, nx);
                    spw = max(spw, nw);
                    if (lane == 0) *(volatile LAS u32x2*)&ctl[C_SPX] = (u32x2){(u32)spx, (u32)spw};
                }
            }
            ++npoll;
            const u32 a = row16_max(~(pf ? max((int)(f - fbase), 0) : NSTEP));
            const int ax = ~max((u32)__builtin_amdgcn_readlane(a, 0), (u32)__builtin_amdgcn_readlane(a, 16));
            const int aw = ~max((u32)__builtin_amdgcn_readlane(a, 32), (u32)__builtin_amdgcn_readlane(a, 48));
            if (ax + aw > sum) {
                sum = ax + aw;
                got = min(ax, aw);
                if (lane == 0) *(volatile LAS u32x2*)&ctl[C_AVX] = (u32x2){(u32)ax, (u32)aw};
            } else if ((npoll & 63) == 0 && (rfl(ld_sys(myws + WS_TMO)) == call || now() - t0 > DEADLINE_TICKS)) { ok = false; break; }
            if (got < 48) __builtin_amdgcn_s_sleep(POLL_SLEEP);
            if (!probe && got >= PROBE_AT) {
                if (rfl(ctl[C_READY + 3]) == NSTEP && rfl(ctl[C_READY + 2]) == NSTEP - 1 && rfl(ctl[C_READY + 1]) == NSTEP - 2 && rfl(ctl[C_READY]) == NSTEP - 3) break;
                __builtin_amdgcn_s_sleep(4);
            }
        }
        if (ok && early) return;
        u32 gmax = 0;
        if (ok) {
            while (rfl(ctl[C_DONE]) < NSTEP) __builtin_amdgcn_s_sleep(1);
            const u64 t1 = now();
            while (now() - t1 < GDELAY) __builtin_amdgcn_s_sleep(1);
            ok = wait_gmax(myws, call, lane, t0, gmax);
        }
        const float gm = __uint_as_float(gmax);
        const float inv = gm > 0.f ? 127.0f * __builtin_amdgcn_rcpf(gm) : 0.f;
        if (lane == 0) {
            *(volatile LAS u32x2*)&ctl[C_INV] = (u32x2){__float_as_uint(inv), ok ? 1u : 2u};
            if (!ok) *(volatile LAS u32x2*)&ctl[C_AVX] = (u32x2){NSTEP, NSTEP};
        }
        if (!ok) raise_tmo(myws, call, lane);
        return;
    }

    const u32 lds0 = (u32)(uintptr_t)(LAS char*)smem;
    if (wave >= 6) {
        __builtin_amdgcn_s_setprio(2);
        const u32x4 srdx = xlocal ? make_srd(p.x, IN_B) : make_srd(myws + WS_X + slot * XSLOT_B, XSLOT_B);
        const u32x4 srdw = wlocal ? make_srd(p.w, IN_B) : make_srd(myws + WS_W + ci * WSLOT_B, WSLOT_B);
        const u32x4 srdb = make_srd(p.x, IN_B);
        const u32 sxs = xlocal ? 128 : 8192, sws = wlocal ? 65536 : 16384;
        const u32 gxs = xlocal ? 32 * KDIM * 2 : 4096, gws = wlocal ? 16384 : 4096;
        u32 vx[4], vw[4], vb[4];
#pragma unroll
        for (int i = 0; i < 4; ++i) {
            vx[i] = xlocal ? xsrc_off(panel, i, lane) - i * 1024 : lane * 16;
            vw[i] = wlocal ? wsrc_off(ci, i, lane) - i * 1024 : lane * 16;
            vb[i] = xsrc_off(panelb, 4 * hb + i, lane) - i * 1024;
        }
        __builtin_amdgcn_s_barrier();
        const int par4 = (rfl(ctl[C_CALL]) & 1) * SPN;
        while ((u32)now() - (u32)t_entry < CDELAY) __builtin_amdgcn_s_sleep(8);
        int sl = wave - 6;
        for (int t = wave - 6; t < NSTEP; t += 4) {
            while (true) {
                const u32x4 d0 = *(volatile LAS u32x4*)&ctl[C_DONE], d1 = *(volatile LAS u32x4*)&ctl[C_DONE + 4];
                const int dmin = min(min(min(d0.x, d0.y), min(d0.z, d0.w)), min(min(d1.x, d1.y), min(d1.z, d1.w)));
                if (rfl(dmin + nstage > t)) break;
                __builtin_amdgcn_s_sleep(1);
            }
            CFENCE();
            const u32 st = lds0 + (u32)sl * stage_b;
            if (dbl) dma4(st + 24576, vb[0], vb[1], vb[2], vb[3], srdb, t * 128);
            const int tx = t < SP0 || xlocal ? t : t + par4, tw = t < SP0 || wlocal ? t : t + par4;
            bool needx = true, needw = true;
            while (true) {
                bool chkx = false, chkw = false;
                while (true) {
                    const u32x4 av = *(volatile LAS u32x4*)&ctl[C_AVX];
                    CFENCE();
                    if (needw && (int)rfl(t < SP0 ? av.y : max(av.y, av.w)) > t) {
                        dma4(st + 8192, vw[0], vw[1], vw[2], vw[3], srdw, tw * sws);
                        dma4(st + 12288, vw[0], vw[1], vw[2], vw[3], srdw, tw * sws + gws);
                        dma4(st + 16384, vw[0], vw[1], vw[2], vw[3], srdw, tw * sws + 2 * gws);
                        dma4(st + 20480, vw[0], vw[1], vw[2], vw[3], srdw, tw * sws + 3 * gws);
                        needw = false;
                        chkw = (int)rfl(av.y) <= t;
                    }
                    if (needx && (int)rfl(t < SP0 ? av.x : max(av.x, av.z)) > t) {
                        dma4(st, vx[0], vx[1], vx[2], vx[3], srdx, tx * sxs);
                        dma4(st + 4096, vx[0], vx[1], vx[2], vx[3], srdx, tx * sxs + gxs);
                        needx = false;
                        chkx = (int)rfl(av.x) <= t;
                    }
                    if (!needx && !needw) break;
                    __builtin_amdgcn_s_sleep(1);
                }
                vmwait<0>();
                if (!chkx && !chkw) break;
                const LAS u32* img = (const LAS u32*)((LAS char*)smem + sl * stage_b) + lane * 16 + (lane >> 1 & 7);
                u32 c[6];
#pragma unroll
                for (int k = 0; k < 6; ++k) c[k] = min(img[k * 1024], img[k * 1024 + 8]);
                const bool zx = chkx && min(c[0], c[1]) == 0, zw = chkw && min(min(c[2], c[3]), min(c[4], c[5])) == 0;
                needx = __any(zx);
                needw = __any(zw);
                if (!needx && !needw) break;
                CFENCE();
                __builtin_amdgcn_s_sleep(RETRY_SLEEP);
            }
            if (lane == 0) ctl[C_READY + (wave - 6)] = t + 1;
            sl += 4;
            if (sl >= nstage) sl -= nstage;
        }
        return;
    }

    const int tb = wave >> 2, wn = wave & 1, wm = tb ? hb : wave >> 1, cw = wm * 2 + wn;
    const int l15 = lane & 15, g4 = lane >> 4;
    u32 ax[2], aw[4];
#pragma unroll
    for (int kk = 0; kk < 2; ++kk) ax[kk] = (tb ? 24576 : 32 * wm * 128) + l15 * 128 + (((4 * kk + g4) ^ (l15 >> 1)) << 4);
    {
        const int q = (lane >> 2) & 3, pp = lane & 3;
#pragma unroll
        for (int nt = 0; nt < 4; ++nt)
            aw[nt] = 8192 + (8 * g4 + q) * 256 + (((8 * wn + 4 * (pp >> 1) + nt) ^ (q | ((g4 & 1) << 3))) << 4) + 8 * (pp & 1);
    }
    u64 o_out = (u64)p.out[(orank ^ rank) & 7];
    u64 o_ws = (u64)p.ws[(orank ^ rank) & 7];
    const u32 inpos = td2 >> tb * 8 & 31;
    asm volatile("" : "+s"(o_out), "+s"(o_ws));
    f32x4 acc[2][4];
#pragma unroll
    for (int mt = 0; mt < 2; ++mt)
#pragma unroll
        for (int nt = 0; nt < 4; ++nt) acc[mt][nt] = (f32x4){0.f, 0.f, 0.f, 0.f};

    __builtin_amdgcn_s_barrier();
    if (tb && !dbl) return;
    const u32x4 k80 = {0x80808080u, 0x80808080u, 0x80808080u, 0x80808080u};
    const u32 inc = td >> (tb ? T_INB : T_INA) & 127;
    auto ri = __builtin_amdgcn_make_buffer_rsrc((void*)o_out, 0, inc ? OUT_B : 0, 0x00020000);
    const u32 io = (u32)((inc & 63) * 64 + cw * 16 + (lane >> 3)) * 512u + ci * 128 + (lane & 7) * 16;
    while ((u32)now() - (u32)t_entry < CDELAY) __builtin_amdgcn_s_sleep(8);
    __builtin_amdgcn_raw_buffer_store_b128(k80, ri, io, 0, 17);
    __builtin_amdgcn_raw_buffer_store_b128(k80, ri, io, 4096, 17);
    const u32 call = rfl(ctl[C_CALL]);
    if (!tb) {
        auto rz = __builtin_amdgcn_make_buffer_rsrc((void*)myws, 0, WS_END, 0x00020000);
#pragma unroll
        for (int k = 0; k < (320 * SPN + 895) / 896; ++k) {
            const u32 id = (b - 32) * 4 + wave + k * 896, nb = SP0 + (~call & 1) * SPN;
            const u32 off = id < 256 * SPN ? WS_X + id / (8 * SPN) * XSLOT_B + nb * 8192 + id % (8 * SPN) * 1024
                                           : WS_W + (id - 256 * SPN) / (16 * SPN) * WSLOT_B + nb * 16384 + id % (16 * SPN) * 1024;
            if (id < 320 * SPN) __builtin_amdgcn_raw_buffer_store_b128((u32x4){0u, 0u, 0u, 0u}, rz, lane * 16, off, 17);
        }
    }

    u64 a1p = (u64)(myws + WS_AMAX1 + ((b & 7) * 128 + (tb ? 28 + (ci & 1) * 2 + (panelb & 1) : (b >> 3) - 4) * 4 + cw) * 128);
    u32 orow = (tb ? rank * 8 + panelb : srank * 8 + panel) * 32768, oslot = ((opos - NEARLY) * 4 + wave) * OTAU;
    asm volatile("" : "+s"(a1p), "+s"(orow), "+s"(oslot));
    s16x8 X[3][2], W[3][4];
#define RDX(set, mt, st, kk) X[set][mt] = *(LAS s16x8*)((st) + ax[kk] + (mt) * 2048)
#define RDW(set, nt, st, kk) W[set][nt] = __builtin_shufflevector( \
        __builtin_amdgcn_ds_read_tr16_b64_v4i16((LAS s16x4*)((st) + aw[nt] + (kk) * 8192)), \
        __builtin_amdgcn_ds_read_tr16_b64_v4i16((LAS s16x4*)((st) + aw[nt] + (kk) * 8192 + 1024)), 0, 1, 2, 3, 4, 5, 6, 7)
#define MM(set, mt, nt) asm volatile("v_mfma_f32_16x16x32_bf16 %0, %1, %2, %0" : "+v"(acc[mt][nt]) : "v"(W[set][nt]), "v"(X[set][mt]))
#define HALF(cur, nxt, st, kk) do { \
        MM(cur, 0, 0); RDX(nxt, 0, st, kk); SCHED(); \
        MM(cur, 0, 1); RDW(nxt, 0, st, kk); SCHED(); \
        MM(cur, 0, 2); RDW(nxt, 1, st, kk); SCHED(); \
        MM(cur, 0, 3); RDW(nxt, 2, st, kk); SCHED(); \
        MM(cur, 1, 0); RDW(nxt, 3, st, kk); SCHED(); \
        MM(cur, 1, 1); RDX(nxt, 1, st, kk); SCHED(); \
        MM(cur, 1, 2); MM(cur, 1, 3); SCHED(); } while (0)
#define WAIT_READY(t) do { u32 v_; while (v_ = ctl[C_READY + ((t) & 3)], (int)rfl(v_) <= (t)) __builtin_amdgcn_s_sleep(1); } while (0)
#define STEP(a, b, c) do { so += stage_b; if (so == nstage * stage_b) so = 0; \
        if (__builtin_expect((int)rfl(rn) <= s + 1, 0)) WAIT_READY(s + 1);        \
        CFENCE(); \
        rn = ctl[C_READY + ((s + 2) & 3)]; \
        LAS char* st_ = (LAS char*)smem + so; \
        HALF(a, c, st_, 0); HALF(b, a, st_, 1); \
        CFENCE(); \
        if (lane == 0) ctl[C_DONE + wave] = s + 2; \
        CFENCE(); ++s; } while (0)
    {
        WAIT_READY(0);
        CFENCE();
        u32 rn = ctl[C_READY + 1], so = 0;
        CFENCE();
        LAS char* st0 = (LAS char*)smem;
        RDX(0, 0, st0, 0); RDW(0, 0, st0, 0); RDW(0, 1, st0, 0); RDW(0, 2, st0, 0); RDW(0, 3, st0, 0); RDX(0, 1, st0, 0);
        SCHED();
        RDX(1, 0, st0, 1); RDW(1, 0, st0, 1); RDW(1, 1, st0, 1); RDW(1, 2, st0, 1); RDW(1, 3, st0, 1); RDX(1, 1, st0, 1);
        CFENCE();
        if (lane == 0) ctl[C_DONE + wave] = 1;
        CFENCE();
#pragma unroll 1
        for (int s = 0; s < NSTEP - 1;) { STEP(0, 1, 2); STEP(2, 0, 1); STEP(1, 2, 0); }
#pragma unroll
        for (int e = 0; e < 16; ++e) MM(e >> 3, (e >> 2) & 1, e & 3);
    }
    asm volatile("" : "+v"(acc[0][0]), "+v"(acc[0][1]), "+v"(acc[0][2]), "+v"(acc[0][3]));
    float m0 = 0.f, m1 = 0.f;
#pragma unroll
    for (int mt = 0; mt < 2; ++mt) {
        if (mt) asm volatile("s_nop 7" : "+v"(m0), "+v"(m1), "+v"(acc[1][0]), "+v"(acc[1][1]), "+v"(acc[1][2]), "+v"(acc[1][3]));
#pragma unroll
        for (int nt = 0; nt < 4; ++nt) {
            m0 = fmaxf(fmaxf(m0, fabsf(acc[mt][nt][0])), fabsf(acc[mt][nt][1]));
            m1 = fmaxf(fmaxf(m1, fabsf(acc[mt][nt][2])), fabsf(acc[mt][nt][3]));
        }
    }
    const u32 muv = wave_max(max(__float_as_uint(m0), __float_as_uint(m1)));
    vmwait<0>();
    if (lane == 0) __hip_atomic_store((GAS u64*)a1p, ((u64)call << 32) | muv, __ATOMIC_RELAXED, __HIP_MEMORY_SCOPE_AGENT);
    const u32 mu = rfl(muv);
    if (early) {
        if (rfl(ctl[C_GRDY]) == 2) return;
        const u32 tn = (u32)now(), ta = (u32)t_entry + TPEND;
        const u32 te = (int)(tn - ta) > 0 ? tn : ta;
        const float rm = mu ? __builtin_amdgcn_rcpf(__uint_as_float(mu)) : 0.f;
        auto re = __builtin_amdgcn_make_buffer_rsrc((void*)o_ws, 0, WS_END, 0x00020000);
        const u32 part = (me * 16 + opos) * 4 + wave;
#pragma unroll
        for (int j = 0; j < 4; ++j) {
            u32x4 o;
#pragma unroll
            for (int e = 0; e < 4; ++e) {
                const f32x4 a = acc[j >> 1][(j & 1) * 2 + (e >> 1)];
                o[e] = __builtin_bit_cast(u32, __builtin_amdgcn_cvt_pknorm_i16(a[(e & 1) * 2] * rm, a[(e & 1) * 2 + 1] * rm));
            }
            const u32 due = te + (((j * 4 * NEARLY + opos * 4 + wave) * ETAU) >> 4);
            while ((int)((u32)now() - due) < 0) __builtin_amdgcn_s_sleep(1);
            __builtin_amdgcn_raw_buffer_store_b128(o, re, lane * 16, WS_E16 + part * 4096 + j * 1024, 17);
        }
        vmwait<0>();
        if (lane == 0) st_sys64((char*)o_ws + WS_EFLAG + part * 128, ((u64)call << 32) | mu);
        return;
    }
    u32 olane = (u32)(32 * wm + l15) * 512u + (wlocal ? ci : me) * 128 + wn * 64 + g4 * 16;
    asm volatile("" : "+v"(olane));
    u32x2 gq;
    while (gq = *(volatile LAS u32x2*)&ctl[C_INV], rfl(gq.y) == 0) {}
    const u32 tq = (u32)now();
    const bool good = rfl(gq.y) == 1;
    const float inv = __uint_as_float(gq.x);
    auto ro = __builtin_amdgcn_make_buffer_rsrc((void*)o_out, 0, OUT_B, 0x00020000);
    auto quant = [&](const f32x4 (&v)[4], float sc) {
        constexpr float MAGIC = 12582912.f;
        u32 dw[4];
#pragma unroll
        for (int nt = 0; nt < 4; ++nt) {
            u32 q[4];
#pragma unroll
            for (int r = 0; r < 4; ++r) q[r] = __float_as_uint(__builtin_amdgcn_fmed3f(__builtin_fmaf(v[nt][r], sc, MAGIC), MAGIC - 127.f, MAGIC + 127.f));
            dw[nt] = __builtin_amdgcn_perm(__builtin_amdgcn_perm(q[3], q[2], 0x0c0c0400u), __builtin_amdgcn_perm(q[1], q[0], 0x0c0c0400u), 0x05040100u);
        }
        const auto a = __builtin_amdgcn_permlane16_swap(dw[0], dw[2], false, false);
        const auto c = __builtin_amdgcn_permlane16_swap(dw[1], dw[3], false, false);
        return (u32x4){a[0], a[1], c[0], c[1]};
    };
#pragma unroll
    for (int mt = 0; mt < 2; ++mt) {
        u32x4 o = quant(acc[mt], inv);
        if (!good) o = k80;
        if (!wlocal) {
            const u32 due = tq + ((oslot + mt * 4 * (29 - NEARLY) * OTAU) >> 4);
            while ((int)((u32)now() - due) < 0) __builtin_amdgcn_s_sleep(1);
        }
        __builtin_amdgcn_raw_buffer_store_b128(o, ro, olane + mt * 16 * 512, orow, 17);
    }
    if (inc && good) {
        const u64 t0 = now();
        if (inpos < NEARLY) {
            const u32 part = (ci * 16 + inpos) * 4 + cw;
            const char* fa = myws + (lane == 0 ? WS_EFLAG + part * 128 : WS_TMO);
            u64 f;
            bool ok = true;
            while (f = ld_sys64(fa), __builtin_amdgcn_readlane((u32)(f >> 32), 0) != call)
                if (__builtin_amdgcn_readlane((u32)f, 1) == call || now() - t0 > DEADLINE_TICKS) { raise_tmo(myws, call, lane); ok = false; break; }
            if (ok && __builtin_amdgcn_readlane((u32)f, 1) != call) {
                CFENCE();
                auto re = __builtin_amdgcn_make_buffer_rsrc((void*)myws, 0, WS_END, 0x00020000);
                u32x4 h[4];
#pragma unroll
                for (int j = 0; j < 4; ++j) h[j] = __builtin_amdgcn_raw_buffer_load_b128(re, lane * 16, WS_E16 + part * 4096 + j * 1024, 17);
                const float sc = __uint_as_float(__builtin_amdgcn_readlane((u32)f, 0)) * (1.0f / 32767.0f) * inv;
#pragma unroll
                for (int mt = 0; mt < 2; ++mt) {
                    f32x4 v[4];
#pragma unroll
                    for (int nt = 0; nt < 4; ++nt) {
                        const u32 lo = h[mt * 2 + (nt >> 1)][(nt & 1) * 2], hi = h[mt * 2 + (nt >> 1)][(nt & 1) * 2 + 1];
                        v[nt] = (f32x4){(float)(short)(lo & 0xffff), (float)((int)lo >> 16), (float)(short)(hi & 0xffff), (float)((int)hi >> 16)};
                    }
                    __builtin_amdgcn_raw_buffer_store_b128(quant(v, sc), ro, olane + mt * 16 * 512, (inc & 63) * 32768, 17);
                }
            }
        } else {
            const u32 due = tq + WLAT + ((((inpos - NEARLY) * 4 + (cw & 2) + 1 + (cw & 1) * 4 * (29 - NEARLY)) * OTAU) >> 4);
            while ((int)((u32)now() - due) < 0) __builtin_amdgcn_s_sleep(1);
            while (true) {
                const u32x4 q0 = __builtin_amdgcn_raw_buffer_load_b128(ri, io, 0, 17), q1 = __builtin_amdgcn_raw_buffer_load_b128(ri, io, 4096, 17);
                const u32 tm = ld_sys(myws + WS_TMO);
                u32 z = 0;
#pragma unroll
                for (int e = 0; e < 4; ++e) z |= (((q0[e] ^ 0x80808080u) - 0x01010101u) & q0[e]) | (((q1[e] ^ 0x80808080u) - 0x01010101u) & q1[e]);
                if (!__any(z & 0x80808080u)) break;
                if (rfl(tm) == call || now() - t0 > DEADLINE_TICKS) { raise_tmo(myws, call, lane); break; }
            }
        }
    }
}

static void build_tiles(int rank, u32* tile, u32* inpos) {
    const int grp = rank >> 2;
    for (int b = 0; b < 32; ++b) tile[b] = 0;
    for (int h = 0; h < 2; ++h) {
        int rows[4][11], nrows[4] = {0, 0, 0, 0}, nn = 0;
        for (int slot = 0; slot < 32; ++slot) {
            const int srank = slot < 20 ? 4 * grp + slot / 5 : 4 * (grp ^ 1) + (slot - 20) / 3;
            const bool special = srank == rank || (slot >= 20 && ((slot - 20) / 3) >> 1 == h);
            const int q = special ? 3 : nn++ / 7;
            rows[q][nrows[q]++] = slot;
        }
        for (int q = 0; q < 4; ++q) {
            int n = 0, nb[2] = {0, 0};
            for (int k = 0; k < nrows[q]; ++k) {
                const int slot = rows[q][k];
                const int srank = slot < 20 ? 4 * grp + slot / 5 : 4 * (grp ^ 1) + (slot - 20) / 3;
                const int panel = slot < 20 ? slot % 5 : 5 + (slot - 20) % 3;
                const u32 base = srank << T_SRANK | panel << T_PANEL | slot << T_SLOT;
                for (int c = 0; c < 2; ++c) {
                    const int ci = 2 * h + c;
                    if (srank != rank) {
                        u32 t = base | 1u << T_WLOCAL | ci << T_CI;
                        if (nb[c] < 4) {
                            t |= 1u << T_DBL | (2 * q + (nb[c] >> 1)) << T_PANELB;
                            if (inpos) inpos[(4 + n) * 8 + 2 * q + h] = (nb[c] & 1) << 16;
                            ++nb[c];
                        }
                        tile[(4 + n++) * 8 + 2 * q + h] = t;
                    }
                    if (srank != 4 * (grp ^ 1) + ci) tile[(4 + n++) * 8 + 2 * q + h] = base | ci << T_CI;
                }
            }
        }
    }
    for (int c = 0; c < 4; ++c) {
        u32 n = 0;
        for (int b = 32; b < NWG; ++b)
            if (!(tile[b] >> T_WLOCAL & 1) && (int)(tile[b] >> T_CI & 3) == c) tile[b] |= n++ << T_INB;
        if (!inpos) continue;
        u32 peer[NWG];
        build_tiles(4 * (grp ^ 1) + c, peer, nullptr);
        int k = 32;
        for (int pass = 0; pass < 2; ++pass)
            for (int b = 32; b < NWG; ++b) {
                if (!(tile[b] >> T_WLOCAL & 1) || (int)(tile[b] >> T_CI & 3) != c) continue;
                if (pass && (!(tile[b] >> T_DBL & 1) || (inpos[b] >> 16 & 1))) continue;
                while (k < NWG && ((peer[k] >> T_WLOCAL & 1) || (int)(peer[k] >> T_CI & 3) != (rank & 3))) ++k;
                if (k == NWG) break;
                const u32 in = 64 | (peer[k] >> T_SRANK & 7) * 8 | (peer[k] >> T_PANEL & 7), pos = peer[k] >> T_INB & 31;
                for (int b2 = b; b2 < (pass ? NWG : b + 1); ++b2)
                    if (b2 == b || ((tile[b2] ^ tile[b]) & (1u << T_WLOCAL | 3u << T_CI | 1u << T_DBL | 7u << T_PANELB)) == 0) {
                        tile[b2] |= in << (pass ? T_INB : T_INA);
                        inpos[b2] |= pos << pass * 8;
                    }
                ++k;
            }
    }
}

extern "C" void kernel_launch(int rank, int nranks, ncclComm_t comm,
                              hipStream_t stream,
                              void* const* d_in, const int* in_sizes, int n_in,
                              void* d_out, int out_size,
                              const int* in_dtypes, int out_dtype,
                              void* const* d_out_all,
                              void* const* d_ws_all, size_t ws_bytes) {
    if (nranks != NR || (unsigned)rank >= (unsigned)NR || n_in != 2 || ws_bytes < WS_END ||
        in_sizes[0] != (int)(IN_B / 2) || in_sizes[1] != (int)(IN_B / 2) || out_size != (int)OUT_B) {
        fprintf(stderr, "kernel_launch: unexpected problem shape\n");
        return;
    }
    Params p;
    p.x = (const char*)d_in[0];
    p.w = (const char*)d_in[1];
    for (int i = 0; i < NR; ++i) { p.ws[i] = (char*)d_ws_all[rank ^ i]; p.out[i] = (char*)d_out_all[rank ^ i]; }
    p.rank = rank;
    p.pad = 0;
    for (int d = 1; d < NR; ++d)
        for (int r = 0; r < 4; ++r)
            for (int jj = 0; jj < 5; ++jj) {
                const u32 t = (4 * jj + r) / 5, u = (4 * jj + r) % 5, me = rank & 3;
                const bool cross = d >= 4;
                if (cross && u >= 3) {
                    p.push[d - 1][r][jj][0] = 0x80000000u | (((rank ^ d) & 3) * 256 + (u - 3) * 4096 + t * 65536);
                    p.push[d - 1][r][jj][1] = WS_W + me * WSLOT_B + t * 16384 + (u - 3) * 1024;
                } else {
                    p.push[d - 1][r][jj][0] = ((cross ? 5 : 0) + u) * 64 * KDIM * 2 + t * 128;
                    p.push[d - 1][r][jj][1] = WS_X + ((cross ? 20 + 3 * me : 5 * me) + u) * XSLOT_B + t * 8192;
                }
            }
    for (int b = 0; b < NWG; ++b) p.inpos[b] = 0;
    build_tiles(rank, p.tile, p.inpos);
    ag_gemm_q8_kernel<<<NWG, NTHREADS, 0, stream>>>(p);
}
```
